```python
import numpy as np

kernel_name = "gpu_a2a_gemm_m4096_k4096_n16384_w8a8_dqsilu"


ABSMAX_RTOL = 0.001

COMPUTE_DTYPE = "int8"
EPILOGUE = "dequant_silu"
IN_QUANT = "int8"
OUT_QUANT = None
IN_DTYPES = {"x": "int8", "w_mat": "int8",
             "scale_x": "float32", "scale_w": "float32"}
WORLD_SIZE = 8

def setup_inputs():

    x = np.random.randint(-127, 128, size=(4096, 4096)).astype(np.int8)
    w_mat = np.random.randint(-127, 128, size=(4096, 16384)).astype(np.int8)
    scale_x = (0.02 * (1.0 + np.random.rand(1))).astype(np.float32)
    scale_w = (0.02 * (1.0 + np.random.rand(1))).astype(np.float32)
    return {"x": x, "w_mat": w_mat, "scale_x": scale_x, "scale_w": scale_w}

def reference(x, w_mat, scale_x, scale_w):
    acc = x.astype(np.float64) @ w_mat.astype(np.float64)
    y = (acc.astype(np.float32) * (scale_x * scale_w)).astype(np.float32)
    return (y / (1.0 + np.exp(-np.clip(y, -60.0, 60.0)))).astype(np.float32)
```

```cpp
#include <hip/hip_runtime.h>
#include <rccl/rccl.h>
#include <cstdio>

#define HIP_CHECK(expr) do { hipError_t _e = (expr); if (_e != hipSuccess) { \
    fprintf(stderr, "HIP error %s at %s:%d\n", hipGetErrorString(_e), __FILE__, __LINE__); \
    return; } } while (0)

#define K_ALT 0
#define K_PBASE 100
#define K_PD0 16
#define K_PD1 25
#define K_POLL0 300
#define K_POLLSLEEP1 16
#define K_POLLSLEEP 8
#define K_WAUX 0
#define K_LDELAY 12
#define K_MAP 0
#define K_WPF 6
#define K_PFDELAY 24
#define K_ABL 0
#define K_LPRIO 0
#define K_XTOUCH 0

namespace {
typedef int v2i __attribute__((ext_vector_type(2)));
typedef int v4i __attribute__((ext_vector_type(4)));
typedef unsigned v4u __attribute__((ext_vector_type(4)));
typedef unsigned v2u __attribute__((ext_vector_type(2)));
#define LAS __attribute__((address_space(3)))
#define GAS __attribute__((address_space(1)))
constexpr int NR = 8;
constexpr int MR = 512, N = 16384, K = 4096, KC = 512;
constexpr int CHUNK = MR * KC;
constexpr unsigned WB = (unsigned)K * N, OB = (unsigned)MR * N * 4;
constexpr int NSTEP = 32;
constexpr int STAGE = 49152, NSLOT = 3, RING = STAGE * NSLOT;
constexpr int C_FULL = RING, C_FREE = RING + 16, C_READY = RING + 32, C_BAD = RING + 36, C_CALL = RING + 40, C_DUMMY = RING + 256, C_TOUCH = RING + 512, LDS_BYTES = RING + 768;
constexpr int GRID = 256;
constexpr size_t WS_TMO = 0, WS_CTR = 65536  , WS_FLAGS = 4096  , WS_STAGE = 1u << 20  ;
constexpr size_t WS_NEED = WS_STAGE + 2ull * NR * CHUNK;
constexpr unsigned long long DEADLINE = 100000000ull;

struct Params { const char* x; const char* w; const float* sx; const float* sw; float* out; char* myws; int rank; char* ws[NR]; };

template <int N_> __device__ __forceinline__ void waitv() { asm volatile("s_waitcnt vmcnt(%0)" :: "n"(N_) : "memory"); }
#define SCHED __builtin_amdgcn_sched_barrier(0)
#define STAMP(j) do {} while (0)
#define STAMPV(j, v) do {} while (0)

__device__ __forceinline__ v4u lds_rd128(unsigned a) { v4u r; asm volatile("ds_read_b128 %0, %1\n\ts_waitcnt lgkmcnt(0)" : "=v"(r) : "v"(a) : "memory"); return r; }
__device__ __forceinline__ unsigned lds_rd32(unsigned a) { unsigned r; asm volatile("ds_read_b32 %0, %1\n\ts_waitcnt lgkmcnt(0)" : "=v"(r) : "v"(a) : "memory"); return r; }
__device__ __forceinline__ void lds_wr32(unsigned a, unsigned v) { asm volatile("ds_write_b32 %0, %1" :: "v"(a), "v"(v) : "memory"); }
__device__ __forceinline__ unsigned umin(unsigned a, unsigned b) { return a < b ? a : b; }
__device__ __forceinline__ unsigned rfl(unsigned v) { return (unsigned)__builtin_amdgcn_readfirstlane((int)v); }

__device__ __forceinline__ float silu_dq(int a, float sc, float sc2) {
  const float f = (float)a;
  const float y = f * sc;
  const float t = __builtin_amdgcn_fmed3f(f * sc2, -86.5617f, 86.5617f);
  const float e = __builtin_amdgcn_exp2f(t);
  return y * __builtin_amdgcn_rcpf(1.0f + e);
}

__device__ __forceinline__ int remote_pos(int i, int swap, int phase) {
  if (!swap) return i;
  if (!phase) return i ^ 1;
  return (i == 0 || i == 27) ? i : (((i - 1) ^ 1) + 1);
}
__device__ __forceinline__ void stage_src(int s, int rank, int swap, int phase, int& src, int& j) {
  if (s < 4) { src = rank; j = s ^ swap; }
  else { const int t = remote_pos(s - 4, swap, phase); j = t / 7; src = (rank + 1 + (t - j * 7)) & 7; }
}

__global__ __launch_bounds__(512, 2) void a2a_gemm(Params p) {
  __shared__ __attribute__((aligned(16))) unsigned char shm[LDS_BYTES];
  LAS unsigned char* lds = (LAS unsigned char*)shm;
  const unsigned lds0 = (unsigned)(size_t)lds;
  const int b = blockIdx.x, xcd = b & 7, idx = b >> 3, mt = idx & 1, nt = K_MAP ? (idx >> 1) * 8 + xcd : xcd * 16 + (idx >> 1);
  const int wid = __builtin_amdgcn_readfirstlane(threadIdx.x >> 6), lane = threadIdx.x & 63;
  const int rank = p.rank;
  char* myws = p.myws;
  const int swap = K_ALT ? mt : 0, phase = (idx >> 1) & 1;

  auto touch_w = [&](int s) {
    int src, j; stage_src(s, rank, swap, phase, src, j);
    auto rw = __builtin_amdgcn_make_buffer_rsrc((void*)p.w, 0, (int)WB, 0x00020000);
    __builtin_amdgcn_raw_ptr_buffer_load_lds(rw, (LAS unsigned*)(lds + C_TOUCH), 4, (unsigned)((lane >> 4) * N + xcd * 2048 + (lane & 15) * 128), (unsigned)((src * 4 + j) * 128 + idx * 4) * (unsigned)N, 0, 0);
  };
  if (wid == 7) {
    __builtin_amdgcn_s_setprio(3);
    const bool pusher = b < 224;
    const int dst = (rank + 1 + b % 7) & 7, u = b / 7, pj = u >> 3, rb = u & 7;
    v4i v[8];
    const unsigned long long t0 = __builtin_amdgcn_s_memrealtime();
    if (pusher) {
      const GAS char* src = (const GAS char*)p.x + (size_t)dst * CHUNK + (rb * 64 + (lane >> 3)) * 512 + pj * 128 + (lane & 7) * 16;
#pragma unroll
      for (int i = 0; i < 8; ++i) v[i] = *(const GAS v4i*)(src + i * 4096);
    }
    GAS unsigned* ctr = (GAS unsigned*)(myws + WS_CTR + b * 128);
    const unsigned done = __hip_atomic_load(ctr, __ATOMIC_RELAXED, __HIP_MEMORY_SCOPE_AGENT);
    asm volatile("s_barrier" ::: "memory");
    const unsigned call = rfl(done) + 1u, par = call & 1u;
    lds_wr32(lds0 + C_CALL, call);
    auto send = [&]() {
      auto rd = __builtin_amdgcn_make_buffer_rsrc((void*)(p.ws[dst] + WS_STAGE + ((size_t)par * NR + rank) * CHUNK), 0, CHUNK, 0x00020000);
      const unsigned off = (unsigned)(pj * 65536 + rb * 8192 + lane * 16);
#pragma unroll
      for (int i = 0; i < 8; ++i) __builtin_amdgcn_raw_buffer_store_b128(v[i], rd, off + i * 1024, 0, 17);
    };
    if (pusher) {
      const unsigned slot_t = pj == 0 ? K_PBASE + rb * K_PD0 : K_PBASE + 8 * K_PD0 + (u - 8) * K_PD1;
      while (__builtin_amdgcn_s_memrealtime() - t0 < (unsigned long long)slot_t) __builtin_amdgcn_s_sleep(1);
      send();
      waitv<0>();
      if (lane == 0) __hip_atomic_store((GAS unsigned*)(p.ws[dst] + WS_FLAGS + ((pj * 8 + rank) * 8 + rb) * 128), call, __ATOMIC_RELAXED, __HIP_MEMORY_SCOPE_SYSTEM);
    }
    if (lane == 0) __hip_atomic_store(ctr, call, __ATOMIC_RELAXED, __HIP_MEMORY_SCOPE_AGENT);
    return;
  }
  if (wid == 6) {
    const unsigned long long tstart = __builtin_amdgcn_s_memrealtime();
    asm volatile("s_barrier" ::: "memory");
    if (K_WPF) {
      __builtin_amdgcn_s_sleep(K_PFDELAY);
      auto rl = __builtin_amdgcn_make_buffer_rsrc((void*)(p.x + (size_t)rank * CHUNK), 0, CHUNK, 0x00020000);
      __builtin_amdgcn_raw_ptr_buffer_load_lds(rl, (LAS unsigned*)(lds + C_TOUCH), 4, (unsigned)(idx * 8192 + lane * 128), 0, 0, 0);
    }
    unsigned call;
    do { __builtin_amdgcn_s_sleep(4); call = rfl(lds_rd32(lds0 + C_CALL)); } while (call == 0u);
    const unsigned par = call & 1u;
    const unsigned long long t0 = __builtin_amdgcn_s_memrealtime();
    GAS unsigned* tmo = (GAS unsigned*)(myws + WS_TMO);
    const GAS unsigned* fbase = (const GAS unsigned*)(myws + WS_FLAGS + (((lane >> 2) & 7) * 8 + mt * 4 + (lane & 3)) * 128);
    const bool own = ((lane >> 2) & 7) == rank || lane >= 32;
    auto rs = __builtin_amdgcn_make_buffer_rsrc((void*)(myws + WS_STAGE + (size_t)par * NR * CHUNK), 0, NR * CHUNK, 0x00020000);
    unsigned ready = 4, avail = 0;
    int j = 0;
    bool seen = own;
    while (__builtin_amdgcn_s_memrealtime() - tstart < (unsigned long long)K_POLL0) __builtin_amdgcn_s_sleep(2);
    for (unsigned polls = 0;; ++polls) {
      if (!seen) seen = __hip_atomic_load(fbase + j * 2048, __ATOMIC_RELAXED, __HIP_MEMORY_SCOPE_SYSTEM) >= call;
      unsigned m = (unsigned)__ballot(seen);
      m &= m >> 1; m &= m >> 2;
      unsigned m8 = 0;
#pragma unroll
      for (int i = 0; i < 8; ++i) m8 |= ((m >> (4 * i)) & 1u) << i;
      avail |= m8 << (8 * j);
      unsigned r = ready;
      while (r < NSTEP) { int src, jj; stage_src((int)r, rank, swap, phase, src, jj); if (!((avail >> (jj * 8 + src)) & 1u)) break; ++r; }
      if (r != ready) {
        ready = r; lds_wr32(lds0 + C_READY, ready | (par << 8));
      }
      if (m8 == 0xffu) {
        if (K_XTOUCH) {
          const unsigned row = (unsigned)(mt * 256 + (idx >> 1) * 16 + (lane & 15));
          const unsigned o0 = (unsigned)(((rank + 1 + (lane >> 4)) & 7) * CHUNK) + row * 128 + j * 65536;
          const unsigned o1 = (lane >> 4) < 3 ? (unsigned)(((rank + 5 + (lane >> 4)) & 7) * CHUNK) + row * 128 + j * 65536 : o0;
          __builtin_amdgcn_raw_ptr_buffer_load_lds(rs, (LAS unsigned*)(lds + C_TOUCH), 4, o0, 0, 0, 17);
          __builtin_amdgcn_raw_ptr_buffer_load_lds(rs, (LAS unsigned*)(lds + C_TOUCH), 4, o1, 0, 0, 17);
        }
        ++j; seen = own;
        if (j == 4) break;
        continue;
      }
      if ((polls & 63u) == 63u && (rfl(__hip_atomic_load(tmo, __ATOMIC_RELAXED, __HIP_MEMORY_SCOPE_AGENT)) == call || __builtin_amdgcn_s_memrealtime() - t0 > DEADLINE)) {
        if (lane == 0) __hip_atomic_store(tmo, call, __ATOMIC_RELAXED, __HIP_MEMORY_SCOPE_AGENT);
        lds_wr32(lds0 + C_BAD, 1u);
        lds_wr32(lds0 + C_READY, NSTEP | (par << 8));
        break;
      }
      if (j == 0) __builtin_amdgcn_s_sleep(K_POLLSLEEP); else __builtin_amdgcn_s_sleep(K_POLLSLEEP1);
    }
    STAMP(5);
    return;
  }
  if (wid >= 4) {
    const int lw = wid - 4, r8 = lane >> 3, c8 = lane & 7;
    const unsigned fulla = lds0 + C_FULL + lw * 4;
    if (K_LPRIO) __builtin_amdgcn_s_setprio(K_LPRIO);
    auto wait_free = [&](unsigned need) {
      for (;;) { const v4u f = lds_rd128(lds0 + C_FREE); if (rfl(umin(umin(f.x, f.y), umin(f.z, f.w))) >= need) break; __builtin_amdgcn_s_sleep(1); }
    };
    auto rw = __builtin_amdgcn_make_buffer_rsrc((void*)p.w, 0, (int)WB, 0x00020000);
    const unsigned wv0 = (unsigned)(r8 * N + nt * 128 + ((c8 ^ (r8 & 6)) * 16)), wv1 = wv0 ^ 16u;
    auto issue_w = [&](int kstep, int slot) {
      const unsigned ws = (unsigned)kstep * 128u * (unsigned)N;
      LAS unsigned char* base = lds + slot * STAGE + 32768;
#pragma unroll
      for (int i = 0; i < 16; ++i)
        __builtin_amdgcn_raw_ptr_buffer_load_lds(rw, (LAS unsigned*)(base + i * 1024), 16, (i & 2) ? wv1 : wv0, ws + i * 8 * N, 0, K_WAUX);
    };
    const unsigned sw0 = (unsigned)((c8 ^ (r8 >> 1)) * 16);
    const unsigned xl0 = (unsigned)((mt * 256 + r8) * 512) + sw0, xl1 = xl0 ^ 64u;
    const unsigned xr0 = (unsigned)((mt * 256 + r8) * 128) + sw0, xr1 = xr0 ^ 64u;
    auto rl = __builtin_amdgcn_make_buffer_rsrc((void*)(p.x + (size_t)rank * CHUNK), 0, CHUNK, 0x00020000);
    auto issue_local = [&](int s, int slot) {
      issue_w(rank * 4 + (s ^ swap), slot);
      LAS unsigned char* base = lds + slot * STAGE;
#pragma unroll
      for (int i = 0; i < 32; ++i)
        __builtin_amdgcn_raw_ptr_buffer_load_lds(rl, (LAS unsigned*)(base + i * 1024), 16, (i & 1) ? xl1 : xl0, (s ^ swap) * 128 + i * 8 * 512, 0, 0);
    };
    auto finish = [&](int s) {
      if (K_WPF && s >= 1 && s <= K_WPF - 3) { touch_w(2 * s + 2); touch_w(2 * s + 3); waitv<2>(); }
      else if (K_WPF && s >= 1 && s + K_WPF < NSTEP) { touch_w(s + K_WPF); waitv<1>(); }
      else waitv<0>();
      lds_wr32(fulla, (unsigned)s + 1u);
    };
    asm volatile("s_barrier" ::: "memory");
    if (lw) __builtin_amdgcn_s_sleep(K_LDELAY);
    issue_local(lw, lw);
    finish(lw);
    if (lw) wait_free(1u);
    issue_local(2 + lw, lw ? 0 : 2);
    finish(2 + lw);
    int slot = 1 + lw; unsigned ready = 4, par = 0;
    for (int s = 4 + lw; s < NSTEP; s += 2) {
      int src, j; stage_src(s, rank, swap, phase, src, j);
      wait_free((unsigned)s - 2u);
      issue_w((K_ABL & 1) ? (s & 1) : src * 4 + j, slot);
      while (ready <= (unsigned)s) {
        const unsigned r = rfl(lds_rd32(lds0 + C_READY));
        ready = r & 255u; par = r >> 8;
        if (ready <= (unsigned)s) __builtin_amdgcn_s_sleep(1);
      }
      auto rs = __builtin_amdgcn_make_buffer_rsrc((void*)(myws + WS_STAGE + (size_t)par * NR * CHUNK), 0, NR * CHUNK, 0x00020000);
      LAS unsigned char* base = lds + slot * STAGE;
      const unsigned so = (K_ABL & 2) ? (unsigned)(((rank + 1) & 7) * CHUNK + (s & 1) * 65536) : (unsigned)(src * CHUNK + j * 65536);
#pragma unroll
      for (int i = 0; i < 8; ++i) {
        __builtin_amdgcn_raw_ptr_buffer_load_lds(rs, (LAS unsigned*)(base + i * 4096), 16, xr0, so + i * 4096, 0, 17);
        __builtin_amdgcn_raw_ptr_buffer_load_lds(rs, (LAS unsigned*)(base + i * 4096), 16, xr1, so + i * 4096, 1024, 17);
        __builtin_amdgcn_raw_ptr_buffer_load_lds(rs, (LAS unsigned*)(base + i * 4096), 16, xr0, so + i * 4096, 2048, 17);
        __builtin_amdgcn_raw_ptr_buffer_load_lds(rs, (LAS unsigned*)(base + i * 4096), 16, xr1, so + i * 4096, 3072, 17);
      }
      finish(s);
      slot = slot == 0 ? 2 : slot - 1;
    }
    return;
  }
  if (wid == 0 && lane < 12) *(volatile LAS unsigned*)(lds + RING + lane * 4) = 0u;
  asm volatile("s_waitcnt lgkmcnt(0)\n\ts_barrier" ::: "memory");
  const int wm = wid >> 1, wn = wid & 1, fr = lane & 15, fq = lane >> 4;
  const unsigned xa0 = (unsigned)(wm * 16384 + fr * 128 + ((fq ^ (fr >> 1)) * 16)), xa1 = xa0 ^ 64u;
  unsigned wa[4];
  { const int r = fr >> 1, pp = fr & 1, g = (r & 6) | (fq & 1);
#pragma unroll
    for (int n = 0; n < 4; ++n) wa[n] = (unsigned)(32768 + (fq * 16 + r) * 128 + (((wn * 4 + n) ^ g) * 16) + pp * 8); }
  volatile LAS unsigned* freep = (volatile LAS unsigned*)(lds + (lane == 0 ? C_FREE + wid * 4 : C_DUMMY + lane * 4));
  unsigned fulloff = C_FULL;
  v4i acc[8][4];
#pragma unroll
  for (int m = 0; m < 8; ++m)
#pragma unroll
    for (int n = 0; n < 4; ++n) { acc[m][n] = (v4i){0, 0, 0, 0}; asm volatile("" : "+v"(acc[m][n])); }
  v4i X0[8], X1[8], W0[4], W1[4];
  auto rdX = [&](v4i& d, unsigned a, int m) { d = *(const LAS v4i*)(lds + a + m * 2048); };
  auto rdW = [&](v4i& d, unsigned a, int kk) {
    v2i lo = __builtin_amdgcn_ds_read_tr8_b64_v2i32((LAS v2i*)(lds + a + kk * 8192));
    v2i hi = __builtin_amdgcn_ds_read_tr8_b64_v2i32((LAS v2i*)(lds + a + kk * 8192 + 1024));
    d = (v4i){lo.x, lo.y, hi.x, hi.y}; };
  auto rd_full = [&]() { return rfl(*(const volatile LAS unsigned*)(lds + fulloff)); };
  auto wait_full = [&](unsigned have, unsigned need) {
    if (__builtin_expect(have < need, 0)) { do { have = rd_full(); } while (have < need); }
    asm volatile("" ::: "memory");
  };
#define READS(XN, WN, xaN, sbN, kkN) do { \
    _Pragma("unroll") for (int n = 0; n < 4; ++n) rdW(WN[n], wa[n] + (sbN), kkN); \
    _Pragma("unroll") for (int m = 0; m < 8; ++m) rdX(XN[m], (xaN) + (sbN), m); } while (0)
#define MMAR(XC, WC, m0, m1) do { \
    _Pragma("unroll") for (int m = m0; m < m1; ++m) _Pragma("unroll") for (int nn = 0; nn < 4; ++nn) { const int n = (m & 1) ? 3 - nn : nn; \
      acc[m][n] = __builtin_amdgcn_mfma_i32_16x16x64_i8(WC[n], XC[m], acc[m][n], 0, 0, 0); } } while (0)
#define MMA(XC, WC) MMAR(XC, WC, 0, 8)
#define SG(mask, n) __builtin_amdgcn_sched_group_barrier(mask, n, 0)
  wait_full(0u, 1u);
  READS(X0, W0, xa0, 0u, 0);
  SCHED;
  unsigned sb = 0;
  for (int s = 0; s < NSTEP - 1; ++s) {
    const unsigned sn = sb == 2 * STAGE ? 0u : sb + STAGE;
    READS(X1, W1, xa1, sb, 1);
    *freep = (unsigned)s + 1u;
    fulloff ^= 4u;
    const unsigned raw = *(const volatile LAS unsigned*)(lds + fulloff);
    MMAR(X0, W0, 0, 6);
#pragma unroll
    for (int i = 0; i < 16; ++i) { SG(0x8, 1); SG(0x100, 1); }
    SG(0x8, 1); SG(0x200, 1); SG(0x8, 5); SG(0x100, 1); SG(0x8, 2);
    SCHED;
    MMAR(X0, W0, 6, 7);
    SCHED;
    const unsigned have = rfl(raw);
    MMAR(X0, W0, 7, 8);
    SG(0x8, 1); SG(0x2, 1); SG(0x8, 3);
    SCHED;
    wait_full(have, (unsigned)s + 2u);
    SCHED;
    READS(X0, W0, xa0, sn, 0);
    MMA(X1, W1);
#pragma unroll
    for (int i = 0; i < 16; ++i) { SG(0x8, 1); SG(0x100, 1); }
    SG(0x8, 16);
    SCHED;
    sb = sn;
  }
  READS(X1, W1, xa1, sb, 1);
  MMA(X0, W0);
#pragma unroll
  for (int i = 0; i < 16; ++i) { SG(0x8, 1); SG(0x100, 1); }
  SG(0x8, 16);
  SCHED;
  MMA(X1, W1);
  SCHED;
  const bool bad = *(volatile LAS unsigned*)(lds + C_BAD) != 0u;
  const float sc = bad ? __builtin_nanf("") : p.sx[0] * p.sw[0], sc2 = sc * -1.44269504f;
  auto ro = __builtin_amdgcn_make_buffer_rsrc((void*)p.out, 0, (int)OB, 0x00020000);
  const unsigned ov = (unsigned)(((mt * 256 + wm * 128 + fr) * N + nt * 128 + wn * 64 + fq * 4) * 4);
#pragma unroll
  for (int m = 0; m < 8; ++m)
#pragma unroll
    for (int n = 0; n < 4; ++n) {
      v4i o;
#pragma unroll
      for (int j = 0; j < 4; ++j) o[j] = __float_as_int(silu_dq(acc[m][n][j], sc, sc2));
      __builtin_amdgcn_raw_buffer_store_b128(o, ro, ov + n * 64, m * 16 * N * 4, 16);
    }
}
}

extern "C" void kernel_launch(int rank, int nranks, ncclComm_t comm,
                              hipStream_t stream,
                              void* const* d_in, const int* in_sizes, int n_in,
                              void* d_out, int out_size,
                              const int* in_dtypes, int out_dtype,
                              void* const* d_out_all,
                              void* const* d_ws_all, size_t ws_bytes) {
  if (nranks != NR || (unsigned)rank >= (unsigned)NR || ws_bytes < WS_NEED || n_in != 4 || out_size != MR * N) {
    fprintf(stderr, "kernel_launch: unexpected problem shape\n");
    return;
  }
  Params p;
  p.x = (const char*)d_in[0]; p.w = (const char*)d_in[1]; p.sx = (const float*)d_in[2]; p.sw = (const float*)d_in[3];
  p.out = (float*)d_out; p.rank = rank; p.myws = (char*)d_ws_all[rank];
  for (int i = 0; i < NR; ++i) p.ws[i] = (char*)d_ws_all[i];
  hipLaunchKernelGGL(a2a_gemm, dim3(GRID), dim3(512), 0, stream, p);
  HIP_CHECK(hipGetLastError());
}
```
